```python
import numpy as np

kernel_name = "gpu_ag_gemm_m4096_k4096_n16384_bf16_gelu"


ABSMAX_RTOL = 0.008

COMPUTE_DTYPE = "bfloat16"
EPILOGUE = "gelu"
IN_QUANT = None
OUT_QUANT = None
IN_DTYPES = {"x": "bfloat16", "w_mat": "bfloat16"}
WORLD_SIZE = 8

def _to_bf16(x):
    u = np.ascontiguousarray(x, dtype=np.float32).view(np.uint32)
    return ((u + 0x7FFF + ((u >> 16) & 1)) & 0xFFFF0000).view(np.float32)

def setup_inputs():

    x = _to_bf16(np.random.randn(4096, 4096).astype(np.float32))
    w_mat = _to_bf16((np.random.randn(4096, 16384) / np.sqrt(4096)).astype(np.float32))
    return {"x": x, "w_mat": w_mat}

def _epilogue(y):
    c = 0.7978845608028654
    return (0.5 * y * (1.0 + np.tanh(c * (y + 0.044715 * y ** 3)))).astype(np.float32)

def reference(x, w_mat):
    y = (x @ w_mat).astype(np.float32)
    return _epilogue(y)
```

```cpp
#include <hip/hip_runtime.h>
#include <rccl/rccl.h>
#include <cstdio>
#include <cstdint>

#define HIP_CHECK(expr) do { hipError_t _e = (expr); if (_e != hipSuccess) { \
    fprintf(stderr, "HIP error %s at %s:%d\n", hipGetErrorString(_e), __FILE__, __LINE__); \
    return; } } while (0)

typedef short bf16x8 __attribute__((ext_vector_type(8)));
typedef short bf16x4 __attribute__((ext_vector_type(4)));
typedef float f32x4 __attribute__((ext_vector_type(4)));
typedef unsigned int u32x4 __attribute__((ext_vector_type(4)));
typedef unsigned int u32x2 __attribute__((ext_vector_type(2)));

constexpr int NRANK = 8, KG = 4096, NL = 2048, MS = 512;
constexpr int BM = 256, BN = 128, BK = 32, NSTEP = KG / BK;
constexpr int A_ST = BM * BK * 2;
constexpr int B_ST = BK * BN * 2;
constexpr int NSTB = 8;
constexpr int PF = 4;
constexpr int EPI_STRIDE = 272;
constexpr int LDS_BYTES = 8 * 32 * EPI_STRIDE;
#define P_NCW 192
#define P_CS 4
constexpr int NCW = P_NCW;
#define P_SLC 8
#define P_SLG 4
#define P_NWH 64
constexpr int NWH = P_NWH;
constexpr int NGW = 256;
constexpr int CS = P_CS;
constexpr int NCH = NSTEP / CS;
constexpr size_t RB_BYTES = (size_t)NSTEP * A_ST;
constexpr size_t XBUF = 16 * RB_BYTES;
constexpr size_t FLAG_OFF = 2 * XBUF;
constexpr size_t FLAG_BYTES = (size_t)16 * NCH * NWH * 4;
constexpr size_t CTR_OFF = FLAG_OFF + ((FLAG_BYTES + 4095) / 4096) * 4096;
constexpr size_t WS_NEED = CTR_OFF + 4096;
constexpr unsigned long long DEADLINE_TICKS = 50000000ull;

struct Args {
  const unsigned short* x;
  const unsigned short* w;
  unsigned short* out;
  char* ws[NRANK];
  int rank;
};

#define WAIT_V(n) asm volatile("s_waitcnt vmcnt(" #n ")" ::: "memory")
#define WAIT_L(n) asm volatile("s_waitcnt lgkmcnt(" #n ")" ::: "memory")
#define BAR() __builtin_amdgcn_s_barrier()

__device__ __forceinline__ unsigned f2bf(float f) {
  unsigned u = __float_as_uint(f);
  return (u + 0x7FFFu + ((u >> 16) & 1u)) >> 16;
}
__device__ __forceinline__ float gelu(float y) {
  const float c2 = 2.0f * 0.7978845608028654f;
  float u = c2 * (y + 0.044715f * y * y * y);
  return y / (1.0f + __expf(-u));
}

__global__ __launch_bounds__(512, 4) void ag_gemm(Args a) {
  extern __shared__ __attribute__((aligned(16))) char shm[];
  const int tid = threadIdx.x;
  char* myws = a.ws[a.rank];
  unsigned* ctr = (unsigned*)(myws + CTR_OFF);
  const unsigned call =
      __hip_atomic_load(ctr, __ATOMIC_RELAXED, __HIP_MEMORY_SCOPE_AGENT) / gridDim.x + 1u;
  const size_t xoff = (call & 1u) ? XBUF : 0;

  if (blockIdx.x < NCW) {
    constexpr int LEN = CS * A_ST / NWH;
    constexpr int NP = LEN / 16;
    constexpr unsigned NITEMS = (unsigned)NCH * 2 * NWH;
    unsigned* tick = ctr + 64;
    const unsigned tbase = (call - 1u) * (NITEMS + NCW);
    unsigned* lt = (unsigned*)shm;
    if (tid == 0) lt[0] = __hip_atomic_fetch_add(tick, 1u, __ATOMIC_RELAXED, __HIP_MEMORY_SCOPE_AGENT) - tbase;
    __syncthreads();
    unsigned item = lt[0];
    int it = 0;
    while (item < NITEMS) {
      unsigned nxt = 0;
      if (tid == 0) nxt = __hip_atomic_fetch_add(tick, 1u, __ATOMIC_RELAXED, __HIP_MEMORY_SCOPE_AGENT) - tbase;
      const unsigned c = item / (2 * NWH), rbl = item & 1, j = (item % (2 * NWH)) >> 1;
      const unsigned rbg = a.rank * 2 + rbl;
      const unsigned short* xs = a.x + (size_t)rbl * BM * KG;
      const unsigned base_o = c * CS * A_ST + j * LEN;
#pragma unroll 4
      for (int idx = tid; idx < NP * NRANK; idx += 512) {
        const int d = idx / NP, piece = idx % NP;
        const unsigned o = base_o + piece * 16;
        const unsigned ks = o >> 14, sub = (o >> 10) & 15, l = (o >> 4) & 63;
        const unsigned row = sub * 16 + (l & 15), k = ks * BK + (l >> 4) * 8;
        const u32x4 v = *(const u32x4*)(xs + (size_t)row * KG + k);
        const unsigned dest = (unsigned)(a.rank + 1 + d) & 7u;
        asm volatile("global_store_dwordx4 %0, %1, off sc0 sc1" :: "v"(a.ws[dest] + xoff + (size_t)rbg * RB_BYTES + o), "v"(v) : "memory");
      }
      ++it;
      if (tid == 0) lt[it & 1] = nxt;
      WAIT_V(0);
      __syncthreads();
      if (tid < NRANK) {
        unsigned* f = (unsigned*)(a.ws[tid] + FLAG_OFF) + ((size_t)rbg * NCH + c) * NWH + j;
        __hip_atomic_store(f, call, __ATOMIC_RELAXED, __HIP_MEMORY_SCOPE_SYSTEM);
      }
      item = lt[it & 1];
    }
  } else {
    const int g = blockIdx.x - NCW;
    const int xcd = g & 7, idx = g >> 3;
    const int rb = (xcd >> 1) * 4 + (idx & 3), cb = (xcd & 1) * 8 + (idx >> 2);
    const int wid = tid >> 6, lane = tid & 63;
    const char* abase = myws + xoff + (size_t)rb * RB_BYTES;
    const unsigned aoff = wid * 2048 + lane * 16;
    const int brow = (tid >> 7) * 8 + ((tid >> 2) & 7);
    const int bch = ((tid >> 5) & 3) * 4 + ((tid & 3) ^ ((brow >> 2) & 3));
    const char* bbase = (const char*)(a.w + cb * BN);
    const unsigned boff = (unsigned)(brow * NL + bch * 8) * 2u;
    const unsigned* flags = (const unsigned*)(myws + FLAG_OFF) + (size_t)rb * NCH * NWH;
    int* badp = (int*)(shm + LDS_BYTES);
    if (tid == 0) *badp = 0;
    int bad = 0;

    auto poll = [&](int c) {
      if (!bad) {
        const unsigned* f = flags + c * NWH + (lane & (NWH - 1));
        unsigned long long t0 = __builtin_amdgcn_s_memrealtime();
        while (true) {
          unsigned v = __hip_atomic_load(f, __ATOMIC_RELAXED, __HIP_MEMORY_SCOPE_SYSTEM);
#pragma unroll
          for (int r = 1; r < NWH / 64; ++r)
            v = min(v, __hip_atomic_load(f + r * 64, __ATOMIC_RELAXED, __HIP_MEMORY_SCOPE_SYSTEM));
          if (__all(v >= call)) break;
          if (__builtin_amdgcn_s_memrealtime() - t0 > DEADLINE_TICKS) { bad = 1; break; }
          __builtin_amdgcn_s_sleep(P_SLG);
        }
      }
    };

    const unsigned lds0 = (unsigned)(size_t)(__attribute__((address_space(3))) char*)shm;
    const int r16 = lane & 15, g4 = lane >> 4;
    const int q = r16 >> 2, p = r16 & 3;
    unsigned ba[2][2];
#pragma unroll
    for (int e = 0; e < 2; ++e)
#pragma unroll
      for (int h = 0; h < 2; ++h) {
        const int row = 8 * g4 + 4 * h + q;
        ba[e][h] = lds0 + 2048 * (row >> 3) + 64 * (row & 7) + 16 * ((e * 2 + (p >> 1)) ^ ((row >> 2) & 3)) + 8 * (p & 1);
      }

    f32x4 acc[2][8];
#pragma unroll
    for (int i = 0; i < 2; ++i)
#pragma unroll
      for (int jn = 0; jn < 8; ++jn) acc[i][jn] = (f32x4){0.f, 0.f, 0.f, 0.f};
    bf16x8 ar[PF][2];

#define ISSUE_A(s_, slot_) do { const char* as_ = abase + (size_t)(s_) * A_ST; \
      asm volatile("global_load_dwordx4 %0, %1, %2" : "=v"(ar[slot_][0]) : "v"(aoff), "s"(as_) : "memory"); \
      asm volatile("global_load_dwordx4 %0, %1, %2 offset:1024" : "=v"(ar[slot_][1]) : "v"(aoff), "s"(as_) : "memory"); } while (0)
#define ISSUE_B(g_) do { _Pragma("unroll") for (int u_ = 0; u_ < PF; ++u_) \
      __builtin_amdgcn_global_load_lds((const unsigned*)(bbase + (size_t)((g_) * PF + u_) * (BK * NL * 2) + boff), \
        (unsigned*)(shm + (((g_) & 1) * PF + u_) * B_ST + tid * 16), 16, 0, 0); } while (0)
#define TRRD(dst_, e_, h_, j_) asm volatile("ds_read_b64_tr_b16 %0, %1 offset:%2" : "=v"(dst_) : "v"(ba[e_][h_]), "n"(j_))
#define RDPAIR(B_, j_) do { TRRD(B_[0], 0, 0, j_); TRRD(B_[1], 0, 1, j_); TRRD(B_[2], 1, 0, j_); TRRD(B_[3], 1, 1, j_); } while (0)
#define WTPAIR(B_, n_) asm volatile("s_waitcnt lgkmcnt(" #n_ ")" : "+v"(B_[0]), "+v"(B_[1]), "+v"(B_[2]), "+v"(B_[3]))
#define MMPAIR(B_, pr_, slot_) do { \
      bf16x8 f0_ = __builtin_shufflevector(B_[0], B_[1], 0, 1, 2, 3, 4, 5, 6, 7); \
      bf16x8 f1_ = __builtin_shufflevector(B_[2], B_[3], 0, 1, 2, 3, 4, 5, 6, 7); \
      acc[0][2 * pr_] = __builtin_amdgcn_mfma_f32_16x16x32_bf16(f0_, ar[slot_][0], acc[0][2 * pr_], 0, 0, 0); \
      acc[1][2 * pr_] = __builtin_amdgcn_mfma_f32_16x16x32_bf16(f0_, ar[slot_][1], acc[1][2 * pr_], 0, 0, 0); \
      acc[0][2 * pr_ + 1] = __builtin_amdgcn_mfma_f32_16x16x32_bf16(f1_, ar[slot_][0], acc[0][2 * pr_ + 1], 0, 0, 0); \
      acc[1][2 * pr_ + 1] = __builtin_amdgcn_mfma_f32_16x16x32_bf16(f1_, ar[slot_][1], acc[1][2 * pr_ + 1], 0, 0, 0); \
      __builtin_amdgcn_sched_barrier(0); } while (0)
#define STEP(u_, LAST_) do { \
      WTPAIR(bx, 4); MMPAIR(bx, 0, u_); RDPAIR(bx, (u_) * B_ST + 1024); \
      WTPAIR(by, 4); MMPAIR(by, 1, u_); RDPAIR(by, (u_) * B_ST + 1536); \
      if (!(LAST_)) { \
        WTPAIR(bx, 4); MMPAIR(bx, 2, u_); RDPAIR(bx, ((u_) + 1) * B_ST); \
        WTPAIR(by, 4); MMPAIR(by, 3, u_); RDPAIR(by, ((u_) + 1) * B_ST + 512); \
      } else { \
        WTPAIR(bx, 4); MMPAIR(bx, 2, u_); WTPAIR(by, 0); MMPAIR(by, 3, u_); \
      } } while (0)
#define WAIT_A(n_, u_) asm volatile("s_waitcnt vmcnt(" #n_ ")" : "+v"(ar[u_][0]), "+v"(ar[u_][1]) :: "memory")

    constexpr int NGRP = NSTEP / PF;
    if (wid == 0) poll(0);
    BAR();
    ISSUE_B(0);
#pragma unroll
    for (int u = 0; u < PF; ++u) ISSUE_A(u, u);

    for (int gq = 0; gq < NGRP - 1; ++gq) {
      WAIT_A(6, 0);
      if (wid == 0 && (((gq + 1) * PF) % CS) == 0) poll((gq + 1) * PF / CS);
      BAR();
      ISSUE_B(gq + 1);
      bf16x4 bx[4], by[4];
      RDPAIR(bx, 0); RDPAIR(by, 512);
      STEP(0, false); ISSUE_A(gq * PF + PF + 0, 0);
      WAIT_A(10, 1); STEP(1, false); ISSUE_A(gq * PF + PF + 1, 1);
      WAIT_A(10, 2); STEP(2, false); ISSUE_A(gq * PF + PF + 2, 2);
      WAIT_A(10, 3); STEP(3, true); ISSUE_A(gq * PF + PF + 3, 3);
      ba[0][0] ^= PF * B_ST; ba[0][1] ^= PF * B_ST; ba[1][0] ^= PF * B_ST; ba[1][1] ^= PF * B_ST;
    }
    {
      WAIT_A(6, 0);
      BAR();
      bf16x4 bx[4], by[4];
      RDPAIR(bx, 0); RDPAIR(by, 512);
      STEP(0, false);
      WAIT_A(4, 1); STEP(1, false);
      WAIT_A(2, 2); STEP(2, false);
      WAIT_A(0, 3); STEP(3, true);
    }

    int tz = threadIdx.x; asm volatile("" : "+v"(tz));
    const int wid2 = tz >> 6, lane2 = tz & 63, r16e = lane2 & 15, g4e = lane2 >> 4;
    if (bad && tz == 0) *badp = 1;
    __syncthreads();
    const bool isbad = *badp != 0;
    char* ep = shm + wid2 * 32 * EPI_STRIDE;
#pragma unroll
    for (int mt = 0; mt < 2; ++mt)
#pragma unroll
      for (int nt = 0; nt < 8; ++nt) {
        u32x2 o;
        o.x = f2bf(gelu(acc[mt][nt][0])) | (f2bf(gelu(acc[mt][nt][1])) << 16);
        o.y = f2bf(gelu(acc[mt][nt][2])) | (f2bf(gelu(acc[mt][nt][3])) << 16);
        if (isbad) { o.x = 0x7FC07FC0u; o.y = 0x7FC07FC0u; }
        *(u32x2*)(ep + (mt * 16 + r16e) * EPI_STRIDE + nt * 32 + g4e * 8) = o;
      }
    unsigned short* obase = a.out + (size_t)(rb * BM + wid2 * 32 + g4e) * NL + cb * BN + r16e * 8;
#pragma unroll
    for (int i = 0; i < 8; ++i) {
      const u32x4 v = *(const u32x4*)(ep + (i * 4 + g4e) * EPI_STRIDE + r16e * 16);
      *(u32x4*)(obase + (size_t)i * 4 * NL) = v;
    }
  }
  if (tid == 0) __hip_atomic_fetch_add(ctr, 1u, __ATOMIC_RELAXED, __HIP_MEMORY_SCOPE_AGENT);
}

extern "C" void kernel_launch(int rank, int nranks, ncclComm_t comm,
                              hipStream_t stream,
                              void* const* d_in, const int* in_sizes, int n_in,
                              void* d_out, int out_size,
                              const int* in_dtypes, int out_dtype,
                              void* const* d_out_all,
                              void* const* d_ws_all, size_t ws_bytes) {
  if (nranks != NRANK || ws_bytes < WS_NEED) {
    fprintf(stderr, "unexpected config nranks=%d ws=%zu\n", nranks, ws_bytes);
    return;
  }
  static bool init = false;
  if (!init) {
    HIP_CHECK(hipFuncSetAttribute((const void*)ag_gemm, hipFuncAttributeMaxDynamicSharedMemorySize,
                                  LDS_BYTES + 16));
    init = true;
  }
  Args a;
  a.x = (const unsigned short*)d_in[0];
  a.w = (const unsigned short*)d_in[1];
  a.out = (unsigned short*)d_out;
  for (int p = 0; p < NRANK; ++p) a.ws[p] = (char*)d_ws_all[p];
  a.rank = rank;
  ag_gemm<<<NCW + NGW, 512, LDS_BYTES + 16, stream>>>(a);
  HIP_CHECK(hipGetLastError());
}
```

```cpp
#include <hip/hip_runtime.h>
#include <rccl/rccl.h>
#include <cstdio>
#include <cstdint>

#define HIP_CHECK(expr) do { hipError_t _e = (expr); if (_e != hipSuccess) { \
    fprintf(stderr, "HIP error %s at %s:%d\n", hipGetErrorString(_e), __FILE__, __LINE__); \
    return; } } while (0)

typedef short bf16x8 __attribute__((ext_vector_type(8)));
typedef short bf16x4 __attribute__((ext_vector_type(4)));
typedef float f32x4 __attribute__((ext_vector_type(4)));
typedef unsigned int u32x4 __attribute__((ext_vector_type(4)));
typedef unsigned int u32x2 __attribute__((ext_vector_type(2)));

constexpr int NRANK = 8, KG = 4096, NL = 2048, MS = 512;
constexpr int BM = 256, BN = 128, BK = 32, NSTEP = KG / BK;
constexpr int A_ST = BM * BK * 2;
constexpr int B_ST = BK * BN * 2;
constexpr int NSTB = 8;
constexpr int PF = 4;
constexpr int EPI_STRIDE = 272;
constexpr int LDS_BYTES = 8 * 32 * EPI_STRIDE;
#define P_NCW 136
#define P_CS 4
constexpr int NCW = P_NCW;
#define P_SLC 8
#define P_SLG 4
#define P_NWH 64
constexpr int NWH = P_NWH;
constexpr int NGW = 256;
constexpr int CS = P_CS;
constexpr int NCH = NSTEP / CS;
constexpr size_t RB_BYTES = (size_t)NSTEP * A_ST;
constexpr size_t XBUF = 16 * RB_BYTES;
constexpr size_t FLAG_OFF = 2 * XBUF;
constexpr size_t FLAG_BYTES = (size_t)16 * NCH * NWH * 4;
constexpr size_t CTR_OFF = FLAG_OFF + ((FLAG_BYTES + 4095) / 4096) * 4096;
constexpr size_t WS_NEED = CTR_OFF + 4096;
constexpr unsigned long long DEADLINE_TICKS = 50000000ull;

struct Args {
  const unsigned short* x;
  const unsigned short* w;
  unsigned short* out;
  char* ws[NRANK];
  int rank;
};

#define WAIT_V(n) asm volatile("s_waitcnt vmcnt(" #n ")" ::: "memory")
#define WAIT_L(n) asm volatile("s_waitcnt lgkmcnt(" #n ")" ::: "memory")
#define BAR() __builtin_amdgcn_s_barrier()

typedef float f32x2 __attribute__((ext_vector_type(2)));
typedef __bf16 bh16x2 __attribute__((ext_vector_type(2)));
__device__ __forceinline__ unsigned gelu2(float y0, float y1) {
  const float k1 = -2.0f * 0.7978845608028654f * 1.4426950408889634f, k2 = k1 * 0.044715f;
  f32x2 y = {y0, y1};
  f32x2 e = y * (y * y * k2 + k1);
  f32x2 r;
  r.x = __builtin_amdgcn_rcpf(1.0f + __builtin_amdgcn_exp2f(e.x));
  r.y = __builtin_amdgcn_rcpf(1.0f + __builtin_amdgcn_exp2f(e.y));
  bh16x2 o = __builtin_convertvector(y * r, bh16x2);
  return __builtin_bit_cast(unsigned, o);
}

__global__ __launch_bounds__(512, 4) void ag_gemm(Args a) {
  extern __shared__ __attribute__((aligned(16))) char shm[];
  const int tid = threadIdx.x;
  char* myws = a.ws[a.rank];
  unsigned* ctr = (unsigned*)(myws + CTR_OFF);
#define GET_CALL() asm volatile("" ::: "memory"); \
  const unsigned call = __hip_atomic_load(ctr, __ATOMIC_RELAXED, __HIP_MEMORY_SCOPE_AGENT) / gridDim.x + 1u; \
  const size_t xoff = (call & 1u) ? XBUF : 0

  if (blockIdx.x < NCW) {
    constexpr int LEN = CS * A_ST / NWH;
    constexpr int NP = LEN / 16;
    constexpr unsigned NITEMS = (unsigned)NCH * 2 * NWH;
    constexpr int NPT = NP * NRANK / 512;
    static_assert(NPT >= 1, "item too small");
    unsigned* tick = ctr + 64;
    unsigned* lt = (unsigned*)shm;
    unsigned item = blockIdx.x;
    u32x4 v[NPT];
    auto load_item = [&](unsigned it_) {
      const unsigned c = it_ / (2 * NWH), rbl = it_ & 1, j = (it_ % (2 * NWH)) >> 1;
      const unsigned short* xs = a.x + (size_t)rbl * BM * KG;
#pragma unroll
      for (int i = 0; i < NPT; ++i) {
        const unsigned o = c * CS * A_ST + j * LEN + ((tid + i * 512) % NP) * 16;
        const unsigned ks = o >> 14, sub = (o >> 10) & 15, l = (o >> 4) & 63;
        const unsigned row = sub * 16 + (l & 15), k = ks * BK + (l >> 4) * 8;
        v[i] = *(const u32x4*)(xs + (size_t)row * KG + k);
      }
    };
    load_item(item);
    GET_CALL();
    const unsigned tbase = (call - 1u) * NITEMS - NCW;
    int it = 0;
    while (true) {
      unsigned nxt = 0;
      if (tid == 0) nxt = __hip_atomic_fetch_add(tick, 1u, __ATOMIC_RELAXED, __HIP_MEMORY_SCOPE_AGENT) - tbase;
      const unsigned c = item / (2 * NWH), rbl = item & 1, j = (item % (2 * NWH)) >> 1;
      const unsigned rbg = a.rank * 2 + rbl;
#pragma unroll
      for (int i = 0; i < NPT; ++i) {
        const int idx = tid + i * 512;
        const unsigned o = c * CS * A_ST + j * LEN + (idx % NP) * 16;
        const unsigned dest = (unsigned)(a.rank + 1 + idx / NP) & 7u;
        asm volatile("global_store_dwordx4 %0, %1, off sc0 sc1" :: "v"(a.ws[dest] + xoff + (size_t)rbg * RB_BYTES + o), "v"(v[i]) : "memory");
      }
      ++it;
      if (tid == 0) lt[it & 1] = nxt;
      WAIT_V(0);
      __syncthreads();
      item = lt[it & 1];
      if (item < NITEMS) load_item(item);
      if (tid < NRANK) {
        unsigned* f = (unsigned*)(a.ws[tid] + FLAG_OFF) + ((size_t)rbg * NCH + c) * NWH + j;
        __hip_atomic_store(f, call, __ATOMIC_RELAXED, __HIP_MEMORY_SCOPE_SYSTEM);
      }
      if (item >= NITEMS) break;
    }
  } else {
    GET_CALL();
    const int g = blockIdx.x - NCW;
    const int xcd = g & 7, idx = g >> 3;
    const int rb = (xcd >> 1) * 4 + (idx & 3), cb = (xcd & 1) * 8 + (idx >> 2);
    const int wid = tid >> 6, lane = tid & 63;
    const char* abase = myws + xoff + (size_t)rb * RB_BYTES;
    const unsigned aoff = wid * 2048 + lane * 16;
    const int brow = (tid >> 7) * 8 + ((tid >> 2) & 7);
    const int bch = ((tid >> 5) & 3) * 4 + ((tid & 3) ^ ((brow >> 2) & 3));
    const char* bbase = (const char*)(a.w + cb * BN);
    const unsigned boff = (unsigned)(brow * NL + bch * 8) * 2u;
    const unsigned* flags = (const unsigned*)(myws + FLAG_OFF) + (size_t)rb * NCH * NWH;
    int* badp = (int*)(shm + LDS_BYTES);
    if (tid == 0) *badp = 0;
    int bad = 0;

    auto poll = [&](int c) {
      if (!bad) {
        const unsigned* f = flags + c * NWH + (lane & (NWH - 1));
        unsigned long long t0 = __builtin_amdgcn_s_memrealtime();
        while (true) {
          unsigned v = __hip_atomic_load(f, __ATOMIC_RELAXED, __HIP_MEMORY_SCOPE_SYSTEM);
#pragma unroll
          for (int r = 1; r < NWH / 64; ++r)
            v = min(v, __hip_atomic_load(f + r * 64, __ATOMIC_RELAXED, __HIP_MEMORY_SCOPE_SYSTEM));
          if (__all(v >= call)) break;
          if (__builtin_amdgcn_s_memrealtime() - t0 > DEADLINE_TICKS) { bad = 1; break; }
          __builtin_amdgcn_s_sleep(P_SLG);
        }
        if (c == 0) { __builtin_amdgcn_fence(__ATOMIC_ACQUIRE, ""); WAIT_V(0); }
        asm volatile("" ::: "memory");
      }
    };

    const unsigned lds0 = (unsigned)(size_t)(__attribute__((address_space(3))) char*)shm;
    const int r16 = lane & 15, g4 = lane >> 4;
    const int q = r16 >> 2, p = r16 & 3;
    unsigned ba[2][2];
#pragma unroll
    for (int e = 0; e < 2; ++e)
#pragma unroll
      for (int h = 0; h < 2; ++h) {
        const int row = 8 * g4 + 4 * h + q;
        ba[e][h] = lds0 + 2048 * (row >> 3) + 64 * (row & 7) + 16 * ((e * 2 + (p >> 1)) ^ ((row >> 2) & 3)) + 8 * (p & 1);
      }

    f32x4 acc[2][8];
#pragma unroll
    for (int i = 0; i < 2; ++i)
#pragma unroll
      for (int jn = 0; jn < 8; ++jn) acc[i][jn] = (f32x4){0.f, 0.f, 0.f, 0.f};
    bf16x8 ar[PF][2];

#define ISSUE_A(s_, slot_) do { const char* as_ = abase + (size_t)(s_) * A_ST; \
      asm volatile("global_load_dwordx4 %0, %1, %2" : "=v"(ar[slot_][0]) : "v"(aoff), "s"(as_) : "memory"); \
      asm volatile("global_load_dwordx4 %0, %1, %2 offset:1024" : "=v"(ar[slot_][1]) : "v"(aoff), "s"(as_) : "memory"); } while (0)
#define ISSUE_B(g_) do { _Pragma("unroll") for (int u_ = 0; u_ < PF; ++u_) \
      __builtin_amdgcn_global_load_lds((const unsigned*)(bbase + (size_t)((g_) * PF + u_) * (BK * NL * 2) + boff), \
        (unsigned*)(shm + (((g_) & 1) * PF + u_) * B_ST + tid * 16), 16, 0, 0); } while (0)
#define TRRD(dst_, e_, h_, j_) asm volatile("ds_read_b64_tr_b16 %0, %1 offset:%2" : "=v"(dst_) : "v"(ba[e_][h_]), "n"(j_))
#define RDPAIR(B_, j_) do { TRRD(B_[0], 0, 0, j_); TRRD(B_[1], 0, 1, j_); TRRD(B_[2], 1, 0, j_); TRRD(B_[3], 1, 1, j_); } while (0)
#define WTPAIR(B_, n_) asm volatile("s_waitcnt lgkmcnt(" #n_ ")" : "+v"(B_[0]), "+v"(B_[1]), "+v"(B_[2]), "+v"(B_[3]))
#define MMPAIR(B_, pr_, slot_) do { \
      bf16x8 f0_ = __builtin_shufflevector(B_[0], B_[1], 0, 1, 2, 3, 4, 5, 6, 7); \
      bf16x8 f1_ = __builtin_shufflevector(B_[2], B_[3], 0, 1, 2, 3, 4, 5, 6, 7); \
      acc[0][2 * pr_] = __builtin_amdgcn_mfma_f32_16x16x32_bf16(f0_, ar[slot_][0], acc[0][2 * pr_], 0, 0, 0); \
      acc[1][2 * pr_] = __builtin_amdgcn_mfma_f32_16x16x32_bf16(f0_, ar[slot_][1], acc[1][2 * pr_], 0, 0, 0); \
      acc[0][2 * pr_ + 1] = __builtin_amdgcn_mfma_f32_16x16x32_bf16(f1_, ar[slot_][0], acc[0][2 * pr_ + 1], 0, 0, 0); \
      acc[1][2 * pr_ + 1] = __builtin_amdgcn_mfma_f32_16x16x32_bf16(f1_, ar[slot_][1], acc[1][2 * pr_ + 1], 0, 0, 0); \
      __builtin_amdgcn_sched_barrier(0); } while (0)
#define STEP(u_, LAST_) do { \
      WTPAIR(bx, 4); MMPAIR(bx, 0, u_); RDPAIR(bx, (u_) * B_ST + 1024); \
      WTPAIR(by, 4); MMPAIR(by, 1, u_); RDPAIR(by, (u_) * B_ST + 1536); \
      if (!(LAST_)) { \
        WTPAIR(bx, 4); MMPAIR(bx, 2, u_); RDPAIR(bx, ((u_) + 1) * B_ST); \
        WTPAIR(by, 4); MMPAIR(by, 3, u_); RDPAIR(by, ((u_) + 1) * B_ST + 512); \
      } else { \
        WTPAIR(bx, 4); MMPAIR(bx, 2, u_); WTPAIR(by, 0); MMPAIR(by, 3, u_); \
      } } while (0)
#define WAIT_A(n_, u_) asm volatile("s_waitcnt vmcnt(" #n_ ")" : "+v"(ar[u_][0]), "+v"(ar[u_][1]) :: "memory")

    constexpr int NGRP = NSTEP / PF;
    if (wid == 0) poll(0);
    BAR();
    ISSUE_B(0);
#pragma unroll
    for (int u = 0; u < PF; ++u) ISSUE_A(u, u);

    for (int gq = 0; gq < NGRP - 1; ++gq) {
      WAIT_A(6, 0);
      if (wid == 0 && (((gq + 1) * PF) % CS) == 0) poll((gq + 1) * PF / CS);
      BAR();
      ISSUE_B(gq + 1);
      bf16x4 bx[4], by[4];
      RDPAIR(bx, 0); RDPAIR(by, 512);
      STEP(0, false); ISSUE_A(gq * PF + PF + 0, 0);
      WAIT_A(10, 1); STEP(1, false); ISSUE_A(gq * PF + PF + 1, 1);
      WAIT_A(10, 2); STEP(2, false); ISSUE_A(gq * PF + PF + 2, 2);
      WAIT_A(10, 3); STEP(3, true); ISSUE_A(gq * PF + PF + 3, 3);
      ba[0][0] ^= PF * B_ST; ba[0][1] ^= PF * B_ST; ba[1][0] ^= PF * B_ST; ba[1][1] ^= PF * B_ST;
    }
    {
      WAIT_A(6, 0);
      BAR();
      bf16x4 bx[4], by[4];
      RDPAIR(bx, 0); RDPAIR(by, 512);
      STEP(0, false);
      WAIT_A(4, 1); STEP(1, false);
      WAIT_A(2, 2); STEP(2, false);
      WAIT_A(0, 3); STEP(3, true);
    }

    int tz = threadIdx.x; asm volatile("" : "+v"(tz));
    const int wid2 = tz >> 6, lane2 = tz & 63, r16e = lane2 & 15, g4e = lane2 >> 4;
    if (bad && tz == 0) *badp = 1;
    __syncthreads();
    const bool isbad = *badp != 0;
    char* ep = shm + wid2 * 32 * EPI_STRIDE;
#pragma unroll
    for (int mt = 0; mt < 2; ++mt)
#pragma unroll
      for (int nt = 0; nt < 8; ++nt) {
        u32x2 o;
        o.x = gelu2(acc[mt][nt][0], acc[mt][nt][1]);
        o.y = gelu2(acc[mt][nt][2], acc[mt][nt][3]);
        if (isbad) { o.x = 0x7FC07FC0u; o.y = 0x7FC07FC0u; }
        *(u32x2*)(ep + (mt * 16 + r16e) * EPI_STRIDE + nt * 32 + g4e * 8) = o;
      }
    unsigned short* obase = a.out + (size_t)(rb * BM + wid2 * 32 + g4e) * NL + cb * BN + r16e * 8;
#pragma unroll
    for (int i = 0; i < 8; ++i) {
      const u32x4 v = *(const u32x4*)(ep + (i * 4 + g4e) * EPI_STRIDE + r16e * 16);
      asm volatile("global_store_dwordx4 %0, %1, off sc0 sc1" :: "v"(obase + (size_t)i * 4 * NL), "v"(v) : "memory");
    }
  }
  if (tid == 0) __hip_atomic_fetch_add(ctr, 1u, __ATOMIC_RELAXED, __HIP_MEMORY_SCOPE_AGENT);
}

extern "C" void kernel_launch(int rank, int nranks, ncclComm_t comm,
                              hipStream_t stream,
                              void* const* d_in, const int* in_sizes, int n_in,
                              void* d_out, int out_size,
                              const int* in_dtypes, int out_dtype,
                              void* const* d_out_all,
                              void* const* d_ws_all, size_t ws_bytes) {
  if (nranks != NRANK || ws_bytes < WS_NEED) {
    fprintf(stderr, "unexpected config nranks=%d ws=%zu\n", nranks, ws_bytes);
    return;
  }
  static bool init = false;
  if (!init) {
    HIP_CHECK(hipFuncSetAttribute((const void*)ag_gemm, hipFuncAttributeMaxDynamicSharedMemorySize,
                                  LDS_BYTES + 16));
    init = true;
  }
  Args a;
  a.x = (const unsigned short*)d_in[0];
  a.w = (const unsigned short*)d_in[1];
  a.out = (unsigned short*)d_out;
  for (int p = 0; p < NRANK; ++p) a.ws[p] = (char*)d_ws_all[p];
  a.rank = rank;
  ag_gemm<<<NCW + NGW, 512, LDS_BYTES + 16, stream>>>(a);
  HIP_CHECK(hipGetLastError());
}
```

```cpp
#include <hip/hip_runtime.h>
#include <rccl/rccl.h>
#include <cstdio>
#include <cstdint>

#define HIP_CHECK(expr) do { hipError_t _e = (expr); if (_e != hipSuccess) { \
    fprintf(stderr, "HIP error %s at %s:%d\n", hipGetErrorString(_e), __FILE__, __LINE__); \
    return; } } while (0)

typedef short bf16x8 __attribute__((ext_vector_type(8)));
typedef short bf16x4 __attribute__((ext_vector_type(4)));
typedef float f32x4 __attribute__((ext_vector_type(4)));
typedef unsigned int u32x4 __attribute__((ext_vector_type(4)));
typedef unsigned int u32x2 __attribute__((ext_vector_type(2)));
typedef unsigned short u16x2 __attribute__((ext_vector_type(2)));

constexpr int NRANK = 8, KG = 4096, NL = 2048, MS = 512;
constexpr int BM = 256, BN = 128, BK = 32, NSTEP = KG / BK;
constexpr int A_ST = BM * BK * 2;
constexpr int B_ST = BK * BN * 2;
constexpr int NSTB = 8;
constexpr int PF = 4;
constexpr int EPI_STRIDE = 272;
constexpr int LDS_BYTES = 8 * 32 * EPI_STRIDE;
#define P_NCW 192
#define P_CS 4
constexpr int NCW = P_NCW;
#define P_SLC 8
#define P_SLG 4
#define P_NWH 64
constexpr int NWH = P_NWH;
constexpr int NGW = 256;
constexpr int CS = P_CS;
constexpr int NCH = NSTEP / CS;
constexpr size_t RB_BYTES = (size_t)NSTEP * A_ST;
constexpr size_t XBUF = 16 * RB_BYTES;
constexpr size_t DEC_OFF = 2 * XBUF;
constexpr size_t FLAG_OFF = 3 * XBUF;
constexpr size_t FLAG_PAR = (size_t)16 * NCH * NWH * 4;
constexpr size_t FLAG2_OFF = FLAG_OFF + 2 * FLAG_PAR;
constexpr size_t FLAG_BYTES = 3 * FLAG_PAR;
#define P_NDW 64
constexpr int NDW = P_NDW;
#define P_TP 4
constexpr int NCC = NCH - P_TP;
#define P_SLD 4
#define P_IPF 1
constexpr int IPF = P_IPF;
constexpr int LDS_TOTAL = LDS_BYTES + 16;
static_assert(NWH == 64 && CS == 4, "one flag per 1 KiB fragment");
constexpr size_t CTR_OFF = FLAG_OFF + ((FLAG_BYTES + 4095) / 4096) * 4096;
constexpr size_t WS_NEED = CTR_OFF + 4096;
constexpr unsigned long long DEADLINE_TICKS = 50000000ull;

struct Args {
  const unsigned short* x;
  const unsigned short* w;
  unsigned short* out;
  char* ws[NRANK];
  int rank;
};

#define WAIT_V(n) asm volatile("s_waitcnt vmcnt(" #n ")" ::: "memory")
#define WAIT_L(n) asm volatile("s_waitcnt lgkmcnt(" #n ")" ::: "memory")
#define BAR() __builtin_amdgcn_s_barrier()

typedef float f32x2 __attribute__((ext_vector_type(2)));
typedef __bf16 bh16x2 __attribute__((ext_vector_type(2)));
__device__ __forceinline__ unsigned gelu2(float y0, float y1) {
  const float k1 = -2.0f * 0.7978845608028654f * 1.4426950408889634f, k2 = k1 * 0.044715f;
  f32x2 y = {y0, y1};
  f32x2 e = y * (y * y * k2 + k1);
  f32x2 r;
  r.x = __builtin_amdgcn_rcpf(1.0f + __builtin_amdgcn_exp2f(e.x));
  r.y = __builtin_amdgcn_rcpf(1.0f + __builtin_amdgcn_exp2f(e.y));
  bh16x2 o = __builtin_convertvector(y * r, bh16x2);
  return __builtin_bit_cast(unsigned, o);
}

#define P_NV 38
__global__ __launch_bounds__(512, 4) __attribute__((amdgpu_num_vgpr(P_NV))) void ag_gemm(Args a) {
  extern __shared__ __attribute__((aligned(16))) char shm[];
  const int tid = threadIdx.x;
  char* myws = a.ws[a.rank];
  unsigned* ctr = (unsigned*)(myws + CTR_OFF);
#define GET_CALL() asm volatile("" ::: "memory"); \
  const unsigned call = __hip_atomic_load(ctr, __ATOMIC_RELAXED, __HIP_MEMORY_SCOPE_AGENT) / gridDim.x + 1u; \
  const size_t xoff = (call & 1u) ? XBUF : 0

  if (blockIdx.x < NCW) {
    constexpr int NI = NWH / IPF;
    constexpr unsigned NITEMS = (unsigned)NCH * 2 * NI;
    unsigned* tick = ctr + 64;
    unsigned* lt = (unsigned*)shm;
    const int wid = tid >> 6, lane = tid & 63;
    char* rp = shm + 1024 + wid * (IPF * 1024);
    unsigned item = blockIdx.x;
    u32x4 v[IPF];
    auto load_item = [&](unsigned it_) {
      const unsigned c = it_ / (2 * NI), rbl = it_ & 1, j = ((it_ % (2 * NI)) >> 1) * IPF;
      const unsigned short* xs = a.x + (size_t)rbl * BM * KG;
#pragma unroll
      for (int i = 0; i < IPF; ++i) {
        const unsigned ks = c * CS + ((j + i) >> 4), sub = (j + i) & 15;
        const unsigned row = sub * 16 + (lane & 15), k = ks * BK + (lane >> 4) * 8;
        v[i] = *(const u32x4*)(xs + (size_t)row * KG + k);
      }
    };
    load_item(item);
    GET_CALL();
    const unsigned tbase = (call - 1u) * NITEMS - NCW;
    const unsigned dest = (unsigned)(a.rank + 1 + wid) & 7u;
    const bool self = wid == NRANK - 1;
    char* dbase = self ? myws + DEC_OFF : a.ws[dest] + xoff;
    int it = 0;
    while (true) {
      unsigned nxt = 0;
      if (tid == 0) nxt = __hip_atomic_fetch_add(tick, 1u, __ATOMIC_RELAXED, __HIP_MEMORY_SCOPE_AGENT) - tbase;
      const unsigned c = item / (2 * NI), rbl = item & 1, j = ((item % (2 * NI)) >> 1) * IPF;
      const unsigned rbg = a.rank * 2 + rbl;
      unsigned meta[IPF];
#pragma unroll
      for (int i = 0; i < IPF; ++i) {
        u16x2 mx, mn;
        {
          const u16x2 a0 = __builtin_bit_cast(u16x2, v[i].x & 0x7FFF7FFFu), a1 = __builtin_bit_cast(u16x2, v[i].y & 0x7FFF7FFFu);
          const u16x2 a2 = __builtin_bit_cast(u16x2, v[i].z & 0x7FFF7FFFu), a3 = __builtin_bit_cast(u16x2, v[i].w & 0x7FFF7FFFu);
          mx = __builtin_elementwise_max(__builtin_elementwise_max(a0, a1), __builtin_elementwise_max(a2, a3));
          mn = __builtin_elementwise_min(__builtin_elementwise_min(a0, a1), __builtin_elementwise_min(a2, a3));
        }
        u16x2 r;
        r.x = mx.x > mx.y ? mx.x : mx.y;
        r.y = (unsigned short)(0x7FFFu - (mn.x < mn.y ? mn.x : mn.y));
#define DPPMAX(ctrl_) r = __builtin_elementwise_max(r, __builtin_bit_cast(u16x2, \
          __builtin_amdgcn_update_dpp(0, __builtin_bit_cast(int, r), ctrl_, 0xF, 0xF, false)))
        DPPMAX(0xB1); DPPMAX(0x4E); DPPMAX(0x141); DPPMAX(0x140);
        unsigned hi = 0, lo = 0;
#pragma unroll
        for (int q = 0; q < 4; ++q) {
          const unsigned rr = __builtin_amdgcn_readlane(__builtin_bit_cast(int, r), q * 16);
          hi = max(hi, rr & 0xFFFFu); lo = max(lo, rr >> 16);
        }
        hi >>= 8; lo = (0x7FFFu - lo) >> 8;
        const unsigned L = hi > 7u ? hi - 7u : 0u;
        char* dst = dbase + (size_t)rbg * RB_BYTES + (size_t)c * CS * A_ST + (j + i) * 1024 + lane * 16;
        const unsigned w0 = __builtin_amdgcn_perm(v[i].y, v[i].x, 0x06040200u), w1 = __builtin_amdgcn_perm(v[i].w, v[i].z, 0x06040200u);
        const unsigned h0 = __builtin_amdgcn_perm(v[i].y, v[i].x, 0x07050301u), h1 = __builtin_amdgcn_perm(v[i].w, v[i].z, 0x07050301u);
        meta[i] = lo >= L ? L : 0xFFu;
        if (self || c >= NCC) {
          asm volatile("global_store_dwordx4 %0, %1, off sc0 sc1" :: "v"(dst), "v"(v[i]) : "memory");
        } else if (lo >= L) {
          const unsigned LL = L * 0x01010101u;
          const unsigned x0 = h0 - LL, x1 = h1 - LL;
          const unsigned nb = ((x0 >> 4) & 0x08080808u) | (x0 & 0x07070707u) | (x1 & 0x80808080u) | ((x1 & 0x07070707u) << 4);
          unsigned* wp = (unsigned*)(rp + i * 1024 + lane * 12);
          wp[0] = w0; wp[1] = w1; wp[2] = nb;
          __builtin_amdgcn_wave_barrier();
          if (lane < 48) {
            const u32x4 t = *(const u32x4*)(rp + i * 1024 + lane * 16);
            asm volatile("global_store_dwordx4 %0, %1, off sc0 sc1" :: "v"(dst), "v"(t) : "memory");
          }
        } else {
          const u32x4 t = {w0, w1, h0, h1};
          asm volatile("global_store_dwordx4 %0, %1, off sc0 sc1" :: "v"(dst), "v"(t) : "memory");
        }
      }
      ++it;
      if (tid == 0) lt[it & 1] = nxt;
      WAIT_V(0);
      __syncthreads();
      item = lt[it & 1];
      if (item < NITEMS) load_item(item);
      if (tid < NRANK * IPF) {
        unsigned m = meta[0];
#pragma unroll
        for (int i = 1; i < IPF; ++i) if ((tid >> 3) == i) m = meta[i];
        const unsigned jf = j + (tid >> 3);
        unsigned* f = (unsigned*)(a.ws[tid & 7] + ((tid & 7) == a.rank ? FLAG2_OFF : FLAG_OFF + ((call & 1u) ? FLAG_PAR : 0))) +
                      ((size_t)rbg * NCH + c) * NWH +
                      ((jf & 15) >> 1) * 8 + (jf >> 4) * 2 + (jf & 1);
        __hip_atomic_store(f, (call << 8) | m, __ATOMIC_RELAXED, __HIP_MEMORY_SCOPE_SYSTEM);
      }
      if (item >= NITEMS) break;
    }
  } else if (blockIdx.x < NCW + NDW) {
    GET_CALL();
    const int wid = __builtin_amdgcn_readfirstlane(tid >> 6), lane = tid & 63;
    const unsigned callm = call & 0xFFFFFFu;
    if (wid == 0) { __builtin_amdgcn_fence(__ATOMIC_ACQUIRE, ""); WAIT_V(0); }
    BAR();
    constexpr int UPC = 14 * 8;
    for (int t = (blockIdx.x - NCW) * 8 + wid; t < NCC * UPC; t += NDW * 8) {
      const int c = t / UPC, r = (t % UPC) >> 3, w = t & 7;
      const int rb = (2 * a.rank + 2 + r) & 15;
      const size_t fo = ((size_t)rb * NCH + c) * NWH + w * 8 + (lane & 7);
      const unsigned* f = (const unsigned*)(myws + FLAG_OFF + ((call & 1u) ? FLAG_PAR : 0)) + fo;
      unsigned fv;
      bool ok = true;
      const unsigned long long t0 = __builtin_amdgcn_s_memrealtime();
      while (true) {
        fv = __hip_atomic_load(f, __ATOMIC_RELAXED, __HIP_MEMORY_SCOPE_SYSTEM);
        if (__all((fv >> 8) == callm)) break;
        if (__builtin_amdgcn_s_memrealtime() - t0 > DEADLINE_TICKS) { ok = false; break; }
        __builtin_amdgcn_s_sleep(P_SLD);
      }
      if (!ok) break;
      asm volatile("" ::: "memory");
      const size_t off = (size_t)rb * RB_BYTES + (size_t)c * CS * A_ST + w * 2048;
      const char* src = myws + xoff + off;
      char* dst = myws + DEC_OFF + off + lane * 16;
      u32x4 d[8];
#pragma unroll
      for (int i = 0; i < 8; ++i) {
        const unsigned m = __builtin_amdgcn_readlane(fv, i) & 0xFFu;
        d[i] = *(const u32x4*)(src + (i >> 1) * A_ST + (i & 1) * 1024 + lane * (m == 0xFFu ? 16 : 12));
      }
#pragma unroll
      for (int i = 0; i < 8; ++i) {
        const unsigned m = __builtin_amdgcn_readlane(fv, i) & 0xFFu;
        unsigned he = d[i].z, ho = d[i].w;
        if (m != 0xFFu) {
          const unsigned ll = m * 0x01010101u, n = d[i].z;
          he = ((n << 4) & 0x80808080u) | ((n & 0x07070707u) + ll);
          ho = (n & 0x80808080u) | (((n >> 4) & 0x07070707u) + ll);
        }
        u32x4 o;
        o.x = __builtin_amdgcn_perm(he, d[i].x, 0x05010400u); o.y = __builtin_amdgcn_perm(he, d[i].x, 0x07030602u);
        o.z = __builtin_amdgcn_perm(ho, d[i].y, 0x05010400u); o.w = __builtin_amdgcn_perm(ho, d[i].y, 0x07030602u);
        asm volatile("global_store_dwordx4 %0, %1, off sc0 sc1" :: "v"(dst + (i >> 1) * A_ST + (i & 1) * 1024), "v"(o) : "memory");
      }
      WAIT_V(0);
      if (lane < 8) __hip_atomic_store((unsigned*)(myws + FLAG2_OFF) + fo, callm << 8, __ATOMIC_RELAXED, __HIP_MEMORY_SCOPE_SYSTEM);
    }
  } else {
    GET_CALL();
    const int g = blockIdx.x - NCW - NDW;
    const int xcd = g & 7, idx = g >> 3;
    const int rb = (xcd >> 1) * 4 + (idx & 3), cb = (xcd & 1) * 8 + (idx >> 2);
    const int wid = __builtin_amdgcn_readfirstlane(tid >> 6), lane = tid & 63;
    const bool own = (rb >> 1) == a.rank;
    const char* abase = myws + DEC_OFF + (size_t)rb * RB_BYTES + wid * 2048;
    const char* abase2 = own ? abase : myws + xoff + (size_t)rb * RB_BYTES + wid * 2048;
    const unsigned aoff = lane * 16;
    const int brow = (tid >> 7) * 8 + ((tid >> 2) & 7);
    const int bch = ((tid >> 5) & 3) * 4 + ((tid & 3) ^ ((brow >> 2) & 3));
    const char* bbase = (const char*)(a.w + cb * BN);
    const unsigned boff = (unsigned)(brow * NL + bch * 8) * 2u;
    const char* flags = myws + FLAG2_OFF + ((size_t)rb * NCH * NWH + wid * 8) * 4;
    const char* flags2 = own ? flags : myws + FLAG_OFF + ((call & 1u) ? FLAG_PAR : 0) + ((size_t)rb * NCH * NWH + wid * 8) * 4;
    const unsigned callm = call & 0xFFFFFFu;
    int* badp = (int*)(shm + LDS_BYTES);
    if (tid == 0) *badp = 0;
    int bad = 0;
    unsigned fv;
#define ISSUE_F(c_) do { unsigned fo_ = aoff; asm volatile("" : "+v"(fo_)); fo_ = (fo_ >> 2) & 28u; \
      asm volatile("global_load_dword v79, %0, %1 sc0 sc1" :: "v"(fo_), "s"(((c_) < NCC ? flags : flags2) + (size_t)(c_) * (NWH * 4)) : "memory"); } while (0)
#define GET_F(n_) asm volatile("s_waitcnt vmcnt(" #n_ ")\n\tv_mov_b32 %0, v79" : "=v"(fv) :: "memory")
    auto check = [&](int c) {
      if (!__all((fv >> 8) == callm) && !bad) {
        unsigned long long t0 = __builtin_amdgcn_s_memrealtime();
        while (true) {
          __builtin_amdgcn_s_sleep(P_SLG);
          ISSUE_F(c);
          GET_F(0);
          if (__all((fv >> 8) == callm)) break;
          if (__builtin_amdgcn_s_memrealtime() - t0 > DEADLINE_TICKS) { bad = 1; break; }
        }
      }
      asm volatile("" ::: "memory");
    };

    const unsigned lds0 = (unsigned)(size_t)(__attribute__((address_space(3))) char*)shm;
    const int r16 = lane & 15, g4 = lane >> 4;
    const int q = r16 >> 2, p = r16 & 3;
    unsigned ba[2][2];
#pragma unroll
    for (int e = 0; e < 2; ++e)
#pragma unroll
      for (int h = 0; h < 2; ++h) {
        const int row = 8 * g4 + 4 * h + q;
        ba[e][h] = lds0 + 2048 * (row >> 3) + 64 * (row & 7) + 16 * ((e * 2 + (p >> 1)) ^ ((row >> 2) & 3)) + 8 * (p & 1);
      }

    f32x4 acc[2][8];
#pragma unroll
    for (int i = 0; i < 2; ++i)
#pragma unroll
      for (int jn = 0; jn < 8; ++jn) acc[i][jn] = (f32x4){0.f, 0.f, 0.f, 0.f};
    asm volatile("" ::: "v76", "v77", "v78", "v79", "v80", "v81", "v82", "v83", "v84", "v85", "v86", "v87", "v88", "v89", "v90", "v91", "v92", "v93", "v94", "v95", "v96", "v97", "v98", "v99", "v100", "v101", "v102", "v103", "v104", "v105", "v106", "v107", "v108", "v109", "v110", "v111", "v112", "v113", "v114", "v115", "v116", "v117", "v118", "v119", "v120", "v121", "v122", "v123", "v124", "v125", "v126", "v127");

#define ISSUE_A(s_, slot_) do { const char* as_ = ((s_) < NCC * CS ? abase : abase2) + (size_t)(s_) * A_ST; \
      asm volatile("global_load_dwordx4 " AR_##slot_##_0 ", %0, %1\n\tglobal_load_dwordx4 " AR_##slot_##_1 ", %0, %1 offset:1024" \
                   :: "v"(aoff), "s"(as_) : "memory"); } while (0)
#define AR_0_0 "v[96:99]"
#define AR_0_1 "v[100:103]"
#define AR_1_0 "v[104:107]"
#define AR_1_1 "v[108:111]"
#define AR_2_0 "v[112:115]"
#define AR_2_1 "v[116:119]"
#define AR_3_0 "v[120:123]"
#define AR_3_1 "v[124:127]"
#define MFMA(c_, b_, slot_, f_) asm volatile("v_mfma_f32_16x16x32_bf16 %0, " b_ ", " AR_##slot_##_##f_ ", %0" : "+v"(c_))
#define ISSUE_B(g_) do { _Pragma("unroll") for (int u_ = 0; u_ < PF; ++u_) \
      asm volatile("s_mov_b32 m0, %0\n\ts_nop 0\n\tglobal_load_lds_dwordx4 %1, %2" \
        :: "s"(lds0 + (((g_) & 1) * PF + u_) * B_ST + wid * 1024), "v"(boff), \
           "s"(bbase + (size_t)((g_) * PF + u_) * (BK * NL * 2)) : "memory"); } while (0)
#define BR_bx_0 "v[80:81]"
#define BR_bx_1 "v[82:83]"
#define BR_bx_2 "v[84:85]"
#define BR_bx_3 "v[86:87]"
#define BF_bx_0 "v[80:83]"
#define BF_bx_1 "v[84:87]"
#define BR_by_0 "v[88:89]"
#define BR_by_1 "v[90:91]"
#define BR_by_2 "v[92:93]"
#define BR_by_3 "v[94:95]"
#define BF_by_0 "v[88:91]"
#define BF_by_1 "v[92:95]"
#define TRRD(dst_, e_, h_, j_) asm volatile("ds_read_b64_tr_b16 " dst_ ", %0 offset:%1" :: "v"(ba[e_][h_]), "n"(j_))
#define RDPAIR(B_, j_) do { TRRD(BR_##B_##_0, 0, 0, j_); TRRD(BR_##B_##_1, 0, 1, j_); TRRD(BR_##B_##_2, 1, 0, j_); TRRD(BR_##B_##_3, 1, 1, j_); } while (0)
#define WTPAIR(B_, n_) asm volatile("s_waitcnt lgkmcnt(" #n_ ")")
#define MMPAIR(B_, pr_, slot_) do { \
      MFMA(acc[0][2 * pr_], BF_##B_##_0, slot_, 0); MFMA(acc[1][2 * pr_], BF_##B_##_0, slot_, 1); \
      MFMA(acc[0][2 * pr_ + 1], BF_##B_##_1, slot_, 0); MFMA(acc[1][2 * pr_ + 1], BF_##B_##_1, slot_, 1); \
      __builtin_amdgcn_sched_barrier(0); } while (0)
#define STEP(u_, LAST_) do { \
      WTPAIR(bx, 4); MMPAIR(bx, 0, u_); RDPAIR(bx, (u_) * B_ST + 1024); \
      WTPAIR(by, 4); MMPAIR(by, 1, u_); RDPAIR(by, (u_) * B_ST + 1536); \
      if (!(LAST_)) { \
        WTPAIR(bx, 4); MMPAIR(bx, 2, u_); RDPAIR(bx, ((u_) + 1) * B_ST); \
        WTPAIR(by, 4); MMPAIR(by, 3, u_); RDPAIR(by, ((u_) + 1) * B_ST + 512); \
      } else { \
        WTPAIR(bx, 4); MMPAIR(bx, 2, u_); WTPAIR(by, 0); MMPAIR(by, 3, u_); \
      } } while (0)

    constexpr int NGRP = NSTEP / PF;
    if (wid == 0) { __builtin_amdgcn_fence(__ATOMIC_ACQUIRE, ""); WAIT_V(0); }
    BAR();
    ISSUE_B(0);
    ISSUE_F(0);
    GET_F(0);
    check(0);
    ISSUE_A(0, 0); ISSUE_A(1, 1); ISSUE_A(2, 2); ISSUE_F(1); ISSUE_A(3, 3);

    for (int gq = 0; gq < NGRP - 1; ++gq) {
      WAIT_V(7);
      BAR();
      ISSUE_B(gq + 1);
      RDPAIR(bx, 0); RDPAIR(by, 512);
      STEP(0, false);
      GET_F(6);
      check(gq + 1);
      ISSUE_A(gq * PF + PF + 0, 0);
      STEP(1, false); ISSUE_A(gq * PF + PF + 1, 1);
      STEP(2, false); ISSUE_A(gq * PF + PF + 2, 2);
      ISSUE_F(gq + 2 < NGRP ? gq + 2 : NGRP - 1);
      WAIT_V(11); STEP(3, true); ISSUE_A(gq * PF + PF + 3, 3);
      ba[0][0] ^= PF * B_ST; ba[0][1] ^= PF * B_ST; ba[1][0] ^= PF * B_ST; ba[1][1] ^= PF * B_ST;
    }
    {
      WAIT_V(7);
      BAR();
      RDPAIR(bx, 0); RDPAIR(by, 512);
      STEP(0, false);
      WAIT_V(5); STEP(1, false);
      WAIT_V(3); STEP(2, false);
      WAIT_V(0); STEP(3, true);
      asm volatile("s_nop 7\n\ts_nop 7\n\ts_nop 7" ::: "memory");
    }

    int tz = threadIdx.x; asm volatile("" : "+v"(tz));
    const int wid2 = tz >> 6, lane2 = tz & 63, r16e = lane2 & 15, g4e = lane2 >> 4;
    if (bad && lane2 == 0) *badp = 1;
    __syncthreads();
    const bool isbad = *badp != 0;
    char* ep = shm + wid2 * 32 * EPI_STRIDE;
#pragma unroll
    for (int mt = 0; mt < 2; ++mt)
#pragma unroll
      for (int nt = 0; nt < 8; ++nt) {
        u32x2 o;
        o.x = gelu2(acc[mt][nt][0], acc[mt][nt][1]);
        o.y = gelu2(acc[mt][nt][2], acc[mt][nt][3]);
        if (isbad) { o.x = 0x7FC07FC0u; o.y = 0x7FC07FC0u; }
        *(u32x2*)(ep + (mt * 16 + r16e) * EPI_STRIDE + nt * 32 + g4e * 8) = o;
      }
    unsigned short* obase = a.out + (size_t)(rb * BM + wid2 * 32 + g4e) * NL + cb * BN + r16e * 8;
#pragma unroll
    for (int i = 0; i < 8; ++i) {
      const u32x4 v = *(const u32x4*)(ep + (i * 4 + g4e) * EPI_STRIDE + r16e * 16);
      asm volatile("global_store_dwordx4 %0, %1, off sc0 sc1" :: "v"(obase + (size_t)i * 4 * NL), "v"(v) : "memory");
    }
  }
  if (tid == 0) __hip_atomic_fetch_add(ctr, 1u, __ATOMIC_RELAXED, __HIP_MEMORY_SCOPE_AGENT);
}

extern "C" void kernel_launch(int rank, int nranks, ncclComm_t comm,
                              hipStream_t stream,
                              void* const* d_in, const int* in_sizes, int n_in,
                              void* d_out, int out_size,
                              const int* in_dtypes, int out_dtype,
                              void* const* d_out_all,
                              void* const* d_ws_all, size_t ws_bytes) {
  if (nranks != NRANK || ws_bytes < WS_NEED) {
    fprintf(stderr, "unexpected config nranks=%d ws=%zu\n", nranks, ws_bytes);
    return;
  }
  static bool init = false;
  if (!init) {
    HIP_CHECK(hipFuncSetAttribute((const void*)ag_gemm, hipFuncAttributeMaxDynamicSharedMemorySize,
                                  LDS_TOTAL));
    init = true;
  }
  Args a;
  a.x = (const unsigned short*)d_in[0];
  a.w = (const unsigned short*)d_in[1];
  a.out = (unsigned short*)d_out;
  for (int p = 0; p < NRANK; ++p) a.ws[p] = (char*)d_ws_all[p];
  a.rank = rank;
  ag_gemm<<<NCW + NDW + NGW, 512, LDS_TOTAL, stream>>>(a);
  HIP_CHECK(hipGetLastError());
}
```
